```python
import numpy as np

kernel_name = "gpu_gemm_rs_m8192_k8192_n8192_f32_gelu"


ABSMAX_RTOL = 0.001

COMPUTE_DTYPE = "float32"
EPILOGUE = "gelu"
IN_QUANT = None
OUT_QUANT = None
IN_DTYPES = {"x": "float32", "w_mat": "float32"}
WORLD_SIZE = 8

def setup_inputs():

    x = np.random.randn(8192, 8192).astype(np.float32)
    w_mat = (np.random.randn(8192, 8192) / np.sqrt(8192)).astype(np.float32)
    return {"x": x, "w_mat": w_mat}

def _epilogue(y):
    c = 0.7978845608028654
    return (0.5 * y * (1.0 + np.tanh(c * (y + 0.044715 * y ** 3)))).astype(np.float32)

def reference(x, w_mat):
    y = (x @ w_mat).astype(np.float32)
    return _epilogue(y)
```

```cpp
#include <hip/hip_runtime.h>
#include <rccl/rccl.h>
#include <cstdio>

#define HIP_CHECK(expr) do { hipError_t _e = (expr); if (_e != hipSuccess) { \
    fprintf(stderr, "HIP error %s at %s:%d\n", hipGetErrorString(_e), __FILE__, __LINE__); \
    return; } } while (0)

namespace gk {
typedef _Float16 h8 __attribute__((ext_vector_type(8)));
typedef float f4 __attribute__((ext_vector_type(4)));
typedef unsigned u32;
#define LAS __attribute__((address_space(3)))

constexpr int NR = 8, GM = 8192, GN = 8192, KS = 1024;
constexpr size_t MiB = 1ull << 20;
constexpr size_t BLK = 32768;
constexpr size_t HBLK = 16384;
constexpr size_t PANEL = 16 * BLK;
constexpr size_t CHUNK = 4 * BLK;
constexpr size_t CTRL_BYTES = 1 * MiB;
constexpr size_t STGX = 0, STGW = 16 * MiB, RCVA = 32 * MiB, RCVB = 64 * MiB, STGO = 128 * MiB, PAR_BYTES = 144 * MiB;
constexpr size_t WS_NEED = CTRL_BYTES + 2 * PAR_BYTES;
constexpr size_t OTILE = 256 * 256 * 4;

constexpr int W_DONE = 0, W_TMO = 32;
constexpr int W_LANDA = 1024;
constexpr int W_LANDB = W_LANDA + 256 * 16;
constexpr int W_SRCF = W_LANDB + 512 * 16;
constexpr int W_OUTF = W_SRCF + 8 * 384;
constexpr int W_PROF = 65536;
constexpr u32 LAND_PER_CALL = 32;

constexpr int GRID = 256, NTHR = 512;
constexpr int B_CONV = 0, N_CONV = 80;
constexpr int B_PULL = 80, PULL_W = 4;
constexpr int B_GEMM = 112, N_GEMM = 128;
constexpr int B_OUTP = 240, OUTP_W = 4;
constexpr int N_CONV_ALL = N_CONV + PULL_W;
constexpr unsigned long long DEADLINE = 50000000ull;
constexpr int LDS_BYTES = 8 * (int)HBLK;

struct Params {
    const float* x; const float* w; float* out;
    char* ws[NR];
    int rank;
};

__device__ __forceinline__ u32 ld_sys(const u32* p) { return __hip_atomic_load(p, __ATOMIC_RELAXED, __HIP_MEMORY_SCOPE_SYSTEM); }
__device__ __forceinline__ u32 ld_agent(const u32* p) { return __hip_atomic_load(p, __ATOMIC_RELAXED, __HIP_MEMORY_SCOPE_AGENT); }
__device__ __forceinline__ void st_sys(u32* p, u32 v) { __hip_atomic_store(p, v, __ATOMIC_RELAXED, __HIP_MEMORY_SCOPE_SYSTEM); }
__device__ __forceinline__ void add_sys(u32* p, u32 v) { (void)__hip_atomic_fetch_add(p, v, __ATOMIC_RELAXED, __HIP_MEMORY_SCOPE_SYSTEM); }
__device__ __forceinline__ void st_wt(void* p, f4 v) {
    asm volatile("global_store_dwordx4 %0, %1, off sc0 sc1\n\ts_nop 1" :: "v"(p), "v"(v) : "memory");
}
__device__ __forceinline__ f4 ld_sys16(const void* p) {
    f4 v; asm volatile("global_load_dwordx4 %0, %1, off sc0 sc1" : "=v"(v) : "v"(p) : "memory"); return v;
}
#define WAIT_V(n) asm volatile("s_waitcnt vmcnt(" #n ")" ::: "memory")
#define WAIT_L(n) asm volatile("s_waitcnt lgkmcnt(" #n ")" ::: "memory")

__device__ __forceinline__ void wait_ge(const u32* flag, u32 target, u32* tmo, u32 call) {
    u32 v = (u32)__builtin_amdgcn_readfirstlane((int)ld_sys(flag));
    if (v >= target) return;
    const unsigned long long t0 = __builtin_amdgcn_s_memrealtime();
    for (;;) {
        __builtin_amdgcn_s_sleep(2);
        v = (u32)__builtin_amdgcn_readfirstlane((int)ld_sys(flag));
        if (v >= target) return;
        if ((u32)__builtin_amdgcn_readfirstlane((int)ld_agent(tmo)) == call || __builtin_amdgcn_s_memrealtime() - t0 > DEADLINE) {
            __hip_atomic_store(tmo, call, __ATOMIC_RELAXED, __HIP_MEMORY_SCOPE_AGENT);
            return;
        }
    }
}

__device__ __forceinline__ float gelu1(float y) {
    const float u = 0.7978845608028654f * (y + 0.044715f * y * y * y);
    const float e = __builtin_amdgcn_exp2f(u * -2.8853900817779268f);
    return y * __builtin_amdgcn_rcpf(1.0f + e);
}

#define STAMP(idx) do {} while (0)

__device__ __forceinline__ int slot_rowblock(int r, int slot) {
    return slot < 4 ? (((((r >> 2) ^ 1) << 2) + slot) * 4 + (r & 3)) : (r * 4 + (slot - 4));
}

__device__ __forceinline__ void run_converter(const Params& P, int cid, u32 call, size_t par) {
    const int r = P.rank, tid = threadIdx.x, wid = tid >> 6, lane = tid & 63;
    char* me = P.ws[r];
    u32* ctrl = (u32*)me;
    for (int it = cid; it < 1024; it += N_CONV_ALL) {
        const int t = it >> 6, j = it & 63;
        if (j < 32) {
            const float* src = P.x + (size_t)(j * 256) * KS + t * 64;
            char* dst = me + par + STGX + (size_t)j * PANEL + (size_t)t * BLK;
            f4 a[4], b[4];
#pragma unroll
            for (int i = 0; i < 4; ++i) {
                const int row = i * 64 + wid * 8 + (lane >> 3), ko = lane & 7;
                const f4* p = (const f4*)(src + (size_t)row * KS + ko * 8);
                a[i] = p[0]; b[i] = p[1];
            }
#pragma unroll
            for (int i = 0; i < 4; ++i) {
                const int row = i * 64 + wid * 8 + (lane >> 3), ko = lane & 7;
                h8 h; h[0] = (_Float16)a[i][0]; h[1] = (_Float16)a[i][1]; h[2] = (_Float16)a[i][2]; h[3] = (_Float16)a[i][3];
                h[4] = (_Float16)b[i][0]; h[5] = (_Float16)b[i][1]; h[6] = (_Float16)b[i][2]; h[7] = (_Float16)b[i][3];
                st_wt(dst + (row >> 7) * HBLK + ko * 2048 + (row & 127) * 16, __builtin_bit_cast(f4, h));
            }
            WAIT_V(0);
            __syncthreads();
            if (tid < 2) {
                const int q = j >> 2;
                const int c = tid == 0 ? q : ((((q >> 2) ^ 1) << 2) + (j & 3));
                const int slot = tid == 0 ? 4 + (j & 3) : (q & 3);
                if (c == r) add_sys(ctrl + W_LANDA + ((slot * 8 + r) * 4 + (t >> 2)) * 16, 8);
                else st_sys((u32*)P.ws[c & 7] + W_SRCF + r * 384 + slot * 16 + t, call);
            }
        } else {
            const int cbg = j - 32;
            const float* src = P.w + (size_t)(t * 64 + wid * 8) * GN + cbg * 256 + lane;
            char* dst = me + par + STGW + (size_t)cbg * PANEL + (size_t)t * BLK;
            float v[4][8];
#pragma unroll
            for (int g = 0; g < 4; ++g)
#pragma unroll
                for (int kk = 0; kk < 8; ++kk) v[g][kk] = src[(size_t)kk * GN + g * 64];
#pragma unroll
            for (int g = 0; g < 4; ++g) {
                h8 h;
#pragma unroll
                for (int kk = 0; kk < 8; ++kk) h[kk] = (_Float16)v[g][kk];
                const int col = g * 64 + lane;
                st_wt(dst + (col >> 7) * HBLK + wid * 2048 + (col & 127) * 16, __builtin_bit_cast(f4, h));
            }
            WAIT_V(0);
            __syncthreads();
            if (tid < 4) {
                const int c = (cbg >> 4) * 4 + tid, cb = cbg & 15;
                if (c == r) add_sys(ctrl + W_LANDB + ((cb * 8 + r) * 4 + (t >> 2)) * 16, 8);
                else st_sys((u32*)P.ws[c & 7] + W_SRCF + r * 384 + 128 + cb * 16 + t, call);
            }
        }
    }
}

struct PItem { const char* src; char* dst; const u32* flag; u32* land; };
__device__ __forceinline__ PItem pull_item(const Params& P, int p, int n, size_t par) {
    const int r = P.rank;
    const int kq = n / 96, m = n - kq * 96, t = kq * 4 + (m & 3), u = m >> 2;
    char* me = P.ws[r]; const char* pe = P.ws[p]; u32* ctrl = (u32*)me;
    PItem it;
    if (u < 8) {
        it.src = pe + par + STGX + (size_t)slot_rowblock(r, u) * PANEL + (size_t)t * BLK;
        it.dst = me + par + RCVA + (size_t)(u * 8 + p) * PANEL + (size_t)t * BLK;
        it.flag = ctrl + W_SRCF + p * 384 + u * 16 + t;
        it.land = ctrl + W_LANDA + ((u * 8 + p) * 4 + kq) * 16;
    } else {
        const int cb = u - 8;
        it.src = pe + par + STGW + (size_t)((r >> 2) * 16 + cb) * PANEL + (size_t)t * BLK;
        it.dst = me + par + RCVB + (size_t)(cb * 8 + p) * PANEL + (size_t)t * BLK;
        it.flag = ctrl + W_SRCF + p * 384 + 128 + cb * 16 + t;
        it.land = ctrl + W_LANDB + ((cb * 8 + p) * 4 + kq) * 16;
    }
    return it;
}
__device__ __forceinline__ void run_in_puller(const Params& P, int link, int g, u32 call, size_t par) {
    const int r = P.rank, tid = threadIdx.x, wid = tid >> 6, lane = tid & 63;
    const int p = (r + 1 + link) & 7;
    u32* ctrl = (u32*)P.ws[r]; u32* tmo = ctrl + W_TMO;
    const int off = wid * 4096 + lane * 16;
    constexpr int NIT = 384;
    int n = g;
    PItem cur = pull_item(P, p, n, par);
    wait_ge(cur.flag, call, tmo, call);
    f4 v0 = ld_sys16(cur.src + off), v1 = ld_sys16(cur.src + off + 1024), v2 = ld_sys16(cur.src + off + 2048), v3 = ld_sys16(cur.src + off + 3072);
    for (;;) {
        const int nn = n + PULL_W;
        const bool more = nn < NIT;
        PItem nxt = cur;
        if (more) { nxt = pull_item(P, p, nn, par); wait_ge(nxt.flag, call, tmo, call); }
        WAIT_V(0);
        asm volatile("" : "+v"(v0), "+v"(v1), "+v"(v2), "+v"(v3));
        st_wt(cur.dst + off, v0); st_wt(cur.dst + off + 1024, v1); st_wt(cur.dst + off + 2048, v2); st_wt(cur.dst + off + 3072, v3);
        if (more) {
            v0 = ld_sys16(nxt.src + off); v1 = ld_sys16(nxt.src + off + 1024); v2 = ld_sys16(nxt.src + off + 2048); v3 = ld_sys16(nxt.src + off + 3072);
            WAIT_V(4);
        } else {
            WAIT_V(0);
        }
        if (lane == 0) add_sys(cur.land, 1);
        if (!more) break;
        cur = nxt; n = nn;
    }
}

__device__ __forceinline__ void run_out_puller(const Params& P, int e, int g, u32 call, size_t par) {
    const int r = P.rank, tid = threadIdx.x, wid = tid >> 6, lane = tid & 63;
    const int p = (((r >> 2) ^ 1) << 2) + e;
    u32* ctrl = (u32*)P.ws[r]; u32* tmo = ctrl + W_TMO;
    const int gw = g * 8 + wid;
    const float qnan = __builtin_nanf("");
    for (int cb = 0; cb < 16; ++cb) {
        wait_ge(ctrl + W_OUTF + (e * 16 + cb) * 32, call, tmo, call);
        const bool bad = (u32)__builtin_amdgcn_readfirstlane((int)ld_agent(tmo)) == call;
        const char* src = P.ws[p] + par + STGO + (size_t)((r & 3) * 16 + cb) * OTILE + (size_t)gw * 8192 + lane * 16;
        float* dst = P.out + (size_t)(e * 256 + gw * 8) * GN + ((p >> 2) * 4096 + cb * 256) + lane * 4;
#pragma unroll
        for (int h = 0; h < 2; ++h) {
            f4 v0, v1, v2, v3;
            if (!bad) {
                v0 = ld_sys16(src + (h * 4 + 0) * 1024); v1 = ld_sys16(src + (h * 4 + 1) * 1024);
                v2 = ld_sys16(src + (h * 4 + 2) * 1024); v3 = ld_sys16(src + (h * 4 + 3) * 1024);
                WAIT_V(0);
                asm volatile("" : "+v"(v0), "+v"(v1), "+v"(v2), "+v"(v3));
            } else { v0 = v1 = v2 = v3 = (f4){qnan, qnan, qnan, qnan}; }
            *(f4*)(dst + (size_t)(h * 4 + 0) * GN) = v0; *(f4*)(dst + (size_t)(h * 4 + 1) * GN) = v1;
            *(f4*)(dst + (size_t)(h * 4 + 2) * GN) = v2; *(f4*)(dst + (size_t)(h * 4 + 3) * GN) = v3;
        }
    }
}

struct Chunk { const char* a; const char* b; const u32* fa; const u32* fb; };
__device__ __forceinline__ Chunk gemm_chunk(const Params& P, int slot, int cb, int c, size_t par) {
    const int r = P.rank;
    int k, kq;
    if (c < 4) { k = r; kq = c; } else { const int z = c - 4; kq = z / 7; k = (r + 1 + (z - kq * 7)) & 7; }
    const char* me = P.ws[r]; const u32* ctrl = (const u32*)me;
    Chunk ch;
    ch.a = me + par + (k == r ? STGX + (size_t)slot_rowblock(r, slot) * PANEL : RCVA + (size_t)(slot * 8 + k) * PANEL) + (size_t)kq * CHUNK;
    ch.b = me + par + (k == r ? STGW + (size_t)((r >> 2) * 16 + cb) * PANEL : RCVB + (size_t)(cb * 8 + k) * PANEL) + (size_t)kq * CHUNK;
    ch.fa = ctrl + W_LANDA + ((slot * 8 + k) * 4 + kq) * 16;
    ch.fb = ctrl + W_LANDB + ((cb * 8 + k) * 4 + kq) * 16;
    return ch;
}

__device__ __forceinline__ void run_gemm(const Params& P, int tile, u32 call, size_t par, LAS unsigned char* lds) {
    const int r = P.rank, tid = threadIdx.x, wid = __builtin_amdgcn_readfirstlane(tid >> 6), lane = tid & 63;
    const int wr = wid >> 2, wc = wid & 3, fr = lane & 15, fq = lane >> 4;
    const int xl = tile & 7, z = tile >> 3, slot = z & 7, cb = xl * 2 + (z >> 3);
    u32* ctrl = (u32*)P.ws[r]; u32* tmo = ctrl + W_TMO;
    const u32 target = LAND_PER_CALL * call;
    const unsigned voff = (unsigned)tid * 16u;
    const unsigned ldsw = (unsigned)wid * 1024u;
    const int aoff = fq * 2048 + (wr * 64 + fr) * 16, boff = fq * 2048 + (wc * 32 + fr) * 16;
#define SA(b, h) (((b) * 2 + (h)) * (int)HBLK)
#define SB(b, h) ((4 + (b) * 2 + (h)) * (int)HBLK)
#define STAGE(bufoff, gbase) do { _Pragma("unroll") for (int _i = 0; _i < 2; ++_i) \
        __builtin_amdgcn_global_load_lds((const unsigned*)((const char*)(gbase) + voff + _i * 8192), (LAS unsigned*)(lds + (bufoff) + ldsw + _i * 8192), 16, 0, 0); } while (0)
#define LDA(dst, b, h) do { _Pragma("unroll") for (int m = 0; m < 4; ++m) _Pragma("unroll") for (int k = 0; k < 2; ++k) dst[m][k] = *(const LAS h8*)(lds + SA(b, h) + aoff + m * 256 + k * 8192); } while (0)
#define LDB(dst, b, h) do { _Pragma("unroll") for (int n = 0; n < 2; ++n) _Pragma("unroll") for (int k = 0; k < 2; ++k) dst[n][k] = *(const LAS h8*)(lds + SB(b, h) + boff + n * 256 + k * 8192); } while (0)
#define MMA(ai, bj, At, Bt) do { __builtin_amdgcn_s_setprio(1); _Pragma("unroll") for (int m = 0; m < 4; ++m) _Pragma("unroll") for (int n = 0; n < 2; ++n) _Pragma("unroll") for (int k = 0; k < 2; ++k) \
        acc[ai][bj][m][n] = __builtin_amdgcn_mfma_f32_16x16x32_f16(Bt[n][k], At[m][k], acc[ai][bj][m][n], 0, 0, 0); __builtin_amdgcn_s_setprio(0); } while (0)
#define BAR __builtin_amdgcn_s_barrier()
#define SCHED __builtin_amdgcn_sched_barrier(0)
#define READY(ch) do { if (wid == 0) { wait_ge((ch).fa, target, tmo, call); wait_ge((ch).fb, target, tmo, call); \
        __builtin_amdgcn_fence(__ATOMIC_ACQUIRE, "agent"); WAIT_V(0); } \
        asm volatile("" ::: "memory"); BAR; asm volatile("" ::: "memory"); } while (0)
    f4 acc[2][2][4][2];
#pragma unroll
    for (int a = 0; a < 2; ++a)
#pragma unroll
        for (int b = 0; b < 2; ++b)
#pragma unroll
            for (int m = 0; m < 4; ++m)
#pragma unroll
                for (int n = 0; n < 2; ++n) acc[a][b][m][n] = (f4){0.f, 0.f, 0.f, 0.f};
    h8 At[4][2], B0[2][2], B1[2][2];
    Chunk cur = gemm_chunk(P, slot, cb, 0, par);
    READY(cur);
    STAMP(1);
    const char* cA = cur.a; const char* cB = cur.b;
    STAGE(SB(0, 0), cB); STAGE(SA(0, 0), cA); STAGE(SB(0, 1), cB + HBLK); STAGE(SA(0, 1), cA + HBLK);
    if (wr == 1) BAR;
    WAIT_V(4); BAR;
    STAGE(SB(1, 0), cB + BLK); STAGE(SA(1, 0), cA + BLK); STAGE(SB(1, 1), cB + BLK + HBLK);
    WAIT_V(6); BAR;
    for (int c = 0; c < 32; ++c) {
        const bool has_next = c + 1 < 32;
        Chunk nxt = gemm_chunk(P, slot, cb, has_next ? c + 1 : c, par);
        const char* nA = nxt.a; const char* nB = nxt.b;
#pragma unroll 1
        for (int t = 0; t < 4; t += 2) {
            const bool last = (t == 2);
            const char* a1 = cA + (size_t)(t + 1) * BLK;
            const char* a2 = last ? nA : cA + (size_t)(t + 2) * BLK; const char* b2 = last ? nB : cB + (size_t)(t + 2) * BLK;
            const char* a3 = a2 + BLK; const char* b3 = b2 + BLK;
            if (last && has_next) READY(nxt);
            LDB(B0, 0, 0); SCHED; LDA(At, 0, 0); STAGE(SA(1, 1), a1 + HBLK);
            WAIT_L(8); BAR; WAIT_L(0); MMA(0, 0, At, B0); BAR; SCHED;
            LDB(B1, 0, 1); STAGE(SB(0, 0), b2);
            BAR; WAIT_L(0); MMA(0, 1, At, B1); BAR;
            LDA(At, 0, 1); STAGE(SA(0, 0), a2);
            BAR; WAIT_L(0); MMA(1, 0, At, B0); BAR; SCHED;
            STAGE(SB(0, 1), b2 + HBLK);
            WAIT_V(6); BAR; MMA(1, 1, At, B1); BAR;
            LDB(B0, 1, 0); SCHED; LDA(At, 1, 0); STAGE(SA(0, 1), a2 + HBLK);
            WAIT_L(8); BAR; WAIT_L(0); MMA(0, 0, At, B0); BAR; SCHED;
            LDB(B1, 1, 1); STAGE(SB(1, 0), b3);
            BAR; WAIT_L(0); MMA(0, 1, At, B1); BAR;
            LDA(At, 1, 1); STAGE(SA(1, 0), a3);
            BAR; WAIT_L(0); MMA(1, 0, At, B0); BAR; SCHED;
            STAGE(SB(1, 1), b3 + HBLK);
            WAIT_V(6); BAR; MMA(1, 1, At, B1); BAR;
        }
        cA = nA; cB = nB;
    }
    WAIT_V(0);
    if (wr == 0) BAR;
    BAR;
    const bool bad = (u32)__builtin_amdgcn_readfirstlane((int)ld_agent(tmo)) == call;
    const float qnan = __builtin_nanf("");
    const int row0 = wr * 64 + fr, col0 = wc * 32 + 4 * fq;
    if (slot >= 4) {
        float* o = P.out + (size_t)((slot - 4) * 256 + row0) * GN + ((r >> 2) * 4096 + cb * 256 + col0);
#pragma unroll
        for (int ai = 0; ai < 2; ++ai)
#pragma unroll
            for (int m = 0; m < 4; ++m)
#pragma unroll
                for (int bj = 0; bj < 2; ++bj)
#pragma unroll
                    for (int n = 0; n < 2; ++n) {
                        f4 v = acc[ai][bj][m][n];
                        v = (f4){gelu1(v[0]), gelu1(v[1]), gelu1(v[2]), gelu1(v[3])};
                        if (bad) v = (f4){qnan, qnan, qnan, qnan};
                        *(f4*)(o + (size_t)(ai * 128 + m * 16) * GN + bj * 128 + n * 16) = v;
                    }
    } else {
        char* o = P.ws[r] + par + STGO + (size_t)(slot * 16 + cb) * OTILE + (size_t)row0 * 1024 + col0 * 4;
#pragma unroll
        for (int ai = 0; ai < 2; ++ai)
#pragma unroll
            for (int m = 0; m < 4; ++m)
#pragma unroll
                for (int bj = 0; bj < 2; ++bj)
#pragma unroll
                    for (int n = 0; n < 2; ++n) {
                        f4 v = acc[ai][bj][m][n];
                        v = (f4){gelu1(v[0]), gelu1(v[1]), gelu1(v[2]), gelu1(v[3])};
                        if (bad) v = (f4){qnan, qnan, qnan, qnan};
                        st_wt(o + (size_t)(ai * 128 + m * 16) * 1024 + (bj * 128 + n * 16) * 4, v);
                    }
        WAIT_V(0);
        __syncthreads();
        if (tid == 0) {
            const int q = (((r >> 2) ^ 1) << 2) + slot;
            st_sys((u32*)P.ws[q & 7] + W_OUTF + ((r & 3) * 16 + cb) * 32, call);
        }
    }
#undef SA
#undef SB
#undef STAGE
#undef LDA
#undef LDB
#undef MMA
#undef READY
}

__global__ void __launch_bounds__(NTHR, 2) fused_kernel(Params P) {
    extern __shared__ __attribute__((aligned(16))) unsigned char shm[];
    u32* ctrl = (u32*)P.ws[P.rank];
    const u32 call = (u32)__builtin_amdgcn_readfirstlane((int)ld_agent(ctrl + W_DONE)) / (u32)GRID + 1u;
    const size_t par = CTRL_BYTES + (size_t)(call & 1u) * PAR_BYTES;
    const int b = blockIdx.x;
    STAMP(0);
    if (b < B_PULL) {
        run_converter(P, b, call, par);
    } else if (b < B_GEMM) {
        const int z = b - B_PULL, label = z & 7, g = z >> 3;
        if (label == 7) run_converter(P, N_CONV + g, call, par);
        else run_in_puller(P, label, g, call, par);
    } else if (b < B_OUTP) {
        run_gemm(P, b - B_GEMM, call, par, (LAS unsigned char*)shm);
    } else {
        const int z = b - B_OUTP;
        run_out_puller(P, z & 3, z >> 2, call, par);
    }
    WAIT_V(0);
    __syncthreads();
    STAMP(63);
    if (threadIdx.x == 0) (void)__hip_atomic_fetch_add(ctrl + W_DONE, 1u, __ATOMIC_RELAXED, __HIP_MEMORY_SCOPE_AGENT);
}
}

extern "C" void kernel_launch(int rank, int nranks, ncclComm_t comm,
                              hipStream_t stream,
                              void* const* d_in, const int* in_sizes, int n_in,
                              void* d_out, int out_size,
                              const int* in_dtypes, int out_dtype,
                              void* const* d_out_all,
                              void* const* d_ws_all, size_t ws_bytes) {
    (void)comm; (void)in_sizes; (void)n_in; (void)out_size; (void)in_dtypes; (void)out_dtype; (void)d_out_all;
    if (nranks != gk::NR || ws_bytes < gk::WS_NEED || (unsigned)rank >= (unsigned)gk::NR) {
        fprintf(stderr, "unsupported configuration\n");
        return;
    }
    static unsigned attr_done = 0;
    int dev = 0;
    HIP_CHECK(hipGetDevice(&dev));
    if (!(attr_done & (1u << (dev & 31)))) {
        HIP_CHECK(hipFuncSetAttribute((const void*)gk::fused_kernel, hipFuncAttributeMaxDynamicSharedMemorySize, gk::LDS_BYTES));
        attr_done |= 1u << (dev & 31);
    }
    gk::Params P;
    P.x = (const float*)d_in[0]; P.w = (const float*)d_in[1]; P.out = (float*)d_out; P.rank = rank;
    for (int p = 0; p < gk::NR; ++p) P.ws[p] = (char*)d_ws_all[p];
    hipLaunchKernelGGL(gk::fused_kernel, dim3(gk::GRID), dim3(gk::NTHR), gk::LDS_BYTES, stream, P);
    HIP_CHECK(hipGetLastError());
}
```

```cpp
#include <hip/hip_runtime.h>
#include <rccl/rccl.h>
#include <cstdio>

#define HIP_CHECK(expr) do { hipError_t _e = (expr); if (_e != hipSuccess) { \
    fprintf(stderr, "HIP error %s at %s:%d\n", hipGetErrorString(_e), __FILE__, __LINE__); \
    return; } } while (0)

namespace gk {
typedef _Float16 h8 __attribute__((ext_vector_type(8)));
typedef float f4 __attribute__((ext_vector_type(4)));
typedef unsigned u32;
#define LAS __attribute__((address_space(3)))

constexpr int PULL_NMAX = 92;
__device__ const unsigned PULL_LEN[8] = {0, 92, 92, 92, 84, 74, 75, 77};
__device__ const unsigned PULL_TAB[8][PULL_NMAX] = {{0},
  {1028,1060,1092,1124,1156,1188,1220,1252,5,37,69,101,1029,1061,1093,1125,1157,1189,1221,1253,1284,1316,1348,1380,6,38,70,102,1030,1062,1094,1126,1158,1190,1222,1254,1285,1317,1349,1381,1412,1444,1476,1508,7,39,71,103,1031,1063,1095,1127,1159,1191,1223,1255,1286,1318,1350,1382,1413,1445,1477,1509,1287,1319,1351,1383,1414,1446,1478,1510,1415,1447,1479,1511,164,132,196,165,133,228,197,166,134,229,198,167,135,230,199,231},
  {1032,1064,1096,1128,1160,1192,1224,1256,9,41,73,105,1033,1065,1097,1129,1161,1193,1225,1257,1288,1320,1352,1384,10,42,74,106,1034,1066,1098,1130,1162,1194,1226,1258,1289,1321,1353,1385,1416,1448,1480,1512,11,43,75,107,1035,1067,1099,1131,1163,1195,1227,1259,1290,1322,1354,1386,1417,1449,1481,1513,1291,1323,1355,1387,1418,1450,1482,1514,1419,1451,1483,1515,136,168,137,232,200,169,138,233,201,170,139,234,202,171,235,203},
  {1036,1068,1100,1132,1164,1196,1228,1260,13,45,77,109,1037,1069,1101,1133,1165,1197,1229,1261,1292,1324,1356,1388,14,46,78,110,1038,1070,1102,1134,1166,1198,1230,1262,1293,1325,1357,1389,1420,1452,1484,1516,15,47,79,111,1039,1071,1103,1135,1167,1199,1231,1263,1294,1326,1358,1390,1421,1453,1485,1517,1295,1327,1359,1391,1422,1454,1486,1518,1423,1455,1487,1519,140,204,172,141,236,205,173,142,237,206,174,143,238,207,175,239},
  {1072,1104,1136,1168,1200,1232,1264,49,81,113,1041,1073,1105,1137,1296,1328,1360,1392,1554,1618,1650,1169,1201,1233,1265,1682,50,82,114,1424,1456,1488,1520,1297,1329,1361,1393,1555,1619,1651,1234,1266,1683,1747,1779,51,83,115,1425,1457,1489,1521,1298,1330,1362,1394,1426,1458,1490,1522,1299,1331,1363,1395,144,176,208,240,1427,1459,1491,1523,145,177,209,241,146,178,210,242,147,179,211,243},
  {1076,1108,1140,1172,1204,1236,1268,21,85,117,1045,1077,1109,1141,1300,1332,1364,1396,1173,1205,1237,1269,22,86,118,1428,1460,1492,1524,1301,1333,1365,1397,1238,1270,23,87,119,1429,1461,1493,1525,1302,1334,1366,1398,1430,1462,1494,1526,1303,1335,1367,1399,148,180,212,244,1431,1463,1495,1527,149,181,213,245,150,182,214,246,151,183,215,247},
  {1080,1112,1144,1176,1208,1240,1272,25,57,121,1049,1081,1113,1145,1304,1336,1368,1400,1177,1209,1241,1273,26,58,122,1723,1432,1464,1496,1528,1305,1337,1369,1401,1242,1274,27,59,123,1433,1465,1497,1529,1306,1338,1370,1402,1434,1466,1498,1530,1307,1339,1371,1403,152,184,216,248,1435,1467,1499,1531,153,185,217,249,154,186,218,250,155,187,219,251},
  {1084,1116,1148,1598,1180,1212,1244,1276,1726,29,61,93,1053,1085,1117,1149,1308,1340,1372,1404,1599,1181,1213,1245,1277,30,62,94,1436,1468,1500,1532,1309,1341,1373,1405,1246,1278,31,63,95,1437,1469,1501,1533,1310,1342,1374,1406,1438,1470,1502,1534,1311,1343,1375,1407,156,188,220,252,1439,1471,1503,1535,157,189,221,253,158,190,222,254,159,191,223,255},
};
constexpr int FWD_LEN = 14;
__device__ const unsigned FWD_TAB[4][14] = {{0},
  {22,86,118,150,23,87,119,151,215,247,191,58,186,59},
  {26,90,122,154,27,91,123,155,219,251,179,54,182,55},
  {30,94,126,158,31,95,127,159,223,255,183,50,178,51},
};
__device__ const unsigned WMASK[4] = {0xc969789au,0xc961b78au,0xc9618b7au,0xc96dc83au};
__device__ const unsigned BMASK[4] = {0x00000000u,0x00000000u,0x00000000u,0x00000000u};
__device__ const unsigned CONV_TAB[256] = {592,1120,2160,4116,8208,16416,32816,580,1140,2148,4136,8248,16436,32804,632,1096,2136,4156,8236,16412,32792,620,1116,2124,61632,128,196,132,200,136,204,140,198,208,144,212,148,216,152,220,17,21,33,156,0,4,8,12,64,84,104,124,160,164,168,172,176,180,184,188,41,53,57,81,97,113,214,29,45,49,61,69,101,117,1,5,9,13,65,85,105,125,129,133,137,141,145,149,153,157,161,165,169,173,177,181,185,189,37,73,89,121,193,2,6,10,14,66,86,106,126,130,134,138,142,146,150,154,158,162,166,170,174,178,182,186,190,25,77,93,109,197,201,3,7,11,15,67,87,107,127,131,135,139,143,147,151,155,159,163,167,171,175,179,183,187,191,205,224,228,232,236,240,244,248,252,209,213,217,221,225,229,233,237,241,245,249,253,194,202,206,210,218,222,226,230,234,238,242,246,250,254,199,195,203,207,211,215,219,223,227,231,235,239,243,247,251,255,18,34,54,58,30,46,82,98,114,50,70,102,118,22,38,74,90,122,26,42,78,94,110,62,19,35,59,47,55,31,51,39,27,83,99,115,23,71,103,119,43,75,91,123,63,79,95,111};

constexpr int NR = 8, GM = 8192, GN = 8192, KS = 1024;
constexpr size_t MiB = 1ull << 20;
constexpr size_t BLK = 32768;
constexpr size_t HBLK = 16384;
constexpr size_t PANEL = 16 * BLK;
constexpr size_t CHUNK = 4 * BLK;
constexpr size_t CTRL_BYTES = 1 * MiB;
constexpr size_t STGX = 0, STGW = 16 * MiB, RCVA = 32 * MiB, RCVB = 64 * MiB, STGO = 128 * MiB, STGP = 144 * MiB, STGH = 160 * MiB, PAR_BYTES = 176 * MiB;
constexpr size_t WS_NEED = CTRL_BYTES + 2 * PAR_BYTES;
constexpr size_t OTILE = 256 * 256 * 4;

constexpr int W_DONE = 0, W_TMO = 32;
constexpr int W_LANDA = 1024;
constexpr int W_LANDB = W_LANDA + 8 * 32;
constexpr int W_FWDC = W_LANDB + 16 * 32;
constexpr int W_LANDP = W_FWDC + 16 * 32;
constexpr int W_PF = W_LANDP + 16 * 32;
constexpr int W_HF = W_PF + 4 * 32;
constexpr int W_RAW = W_HF + 1024;
constexpr int W_SRCF = W_RAW + 32 * 32;
constexpr int W_OUTF = W_SRCF + 8 * 384;
constexpr int W_HEAD = W_OUTF + 4 * 32;
constexpr int W_ORDER = W_HEAD + 4 * 16 * 32;
constexpr int W_PROF = 65536;
constexpr u32 LAND_PER_CALL = 32;

constexpr int GRID = 256, NTHR = 512;
#define GK_PWG 5
#define GK_DWG 2
constexpr int PULL_W = GK_PWG, DYN_W = GK_DWG;
constexpr int N_GEMM = 192;
#define GK_GSLEEP 32
#define GK_LAYOUT 7
enum { R_CONV = 0, R_STATIC = 1, R_DYN = 2, R_SEQ = 3, R_GEMM = 4, R_WATCH = 5 };
struct RoleTab { unsigned role[GRID]; };
constexpr RoleTab make_roles() {
    RoleTab t{}; int nlast = GRID - 8 * PULL_W - 1 - 64, nfirst = 0;
    for (int b = 0; b < GRID; ++b) {
        const int label = b & 7, j = b >> 3; int role = R_CONV, arg = 0;
        if (j < PULL_W) role = R_STATIC;
        else if (j < PULL_W + 24) {
            const int z = j - PULL_W; role = R_GEMM;
            if (GK_LAYOUT == 7) {
                if (z < 8) arg = (z & 3) * 16 + label * 2 + (z >> 2);
                else arg = (1 + (label >> 2)) * 64 + (z & 3) * 16 + (label & 3) * 4 + ((z - 8) >> 2);
            } else if (label < 4 && z < 16) arg = (z & 3) * 16 + label * 4 + (z >> 2);
            else {
                const int col = label < 4 ? 24 + label * 2 + ((z - 16) >> 2) : (label - 4) * 6 + (z >> 2);
                arg = (1 + (col >> 4)) * 64 + (z & 3) * 16 + (col & 15);
            }
        }
        else if (j < PULL_W + 24 + DYN_W) role = R_DYN;
        else if (j == PULL_W + 24 + DYN_W && label == 0) role = R_SEQ;
        else if (j == PULL_W + 24 + DYN_W && label == 1) role = R_WATCH;
        int cid = 0;
        if (role == R_GEMM && arg < 64) cid = nlast++; else if (role != R_STATIC && role != R_WATCH) cid = nfirst++;
        t.role[b] = (unsigned)role | ((unsigned)arg << 8) | ((unsigned)cid << 16);
    }
    return t;
}
static_assert(PULL_W + 24 + DYN_W < 32, "roles must fit the 32 workgroups of an XCD");
constexpr int LOG_OF[8] = {0, 1, 2, 6, 3, 7, 4, 5};
constexpr int PHYS_OF[8] = {0, 1, 2, 4, 6, 7, 3, 5};
constexpr int XLINK_PHYS[8][8] = {
    {2, 3, 6, 0, 1, 4, 5, 7}, {4, 5, 6, 0, 1, 2, 3, 7}, {1, 2, 4, 0, 3, 5, 6, 7}, {1, 3, 5, 0, 2, 4, 6, 7},
    {1, 3, 5, 0, 2, 4, 6, 7}, {1, 2, 4, 0, 3, 5, 6, 7}, {4, 5, 6, 0, 1, 2, 3, 7}, {2, 3, 6, 0, 1, 4, 5, 7}};
struct LinkTab { unsigned link[8][8]; unsigned phys[8]; };
constexpr LinkTab make_links() {
    LinkTab t{};
    for (int l = 0; l < 8; ++l) { t.phys[l] = (unsigned)PHYS_OF[l];
        for (int x = 0; x < 8; ++x) t.link[l][x] = (unsigned)(l ^ LOG_OF[PHYS_OF[l] ^ XLINK_PHYS[PHYS_OF[l]][x]]); }
    return t;
}
__device__ const LinkTab LINKS = make_links();
__device__ const RoleTab ROLES = make_roles();
constexpr int N_CONV_ALL = GRID - 8 * PULL_W - 1;
constexpr unsigned long long DEADLINE = 30000000ull;
constexpr int LDS_STAGE = 8 * (int)HBLK;
constexpr int LDS_BYTES = LDS_STAGE + 64;

struct Params {
    const float* x; const float* w; float* out;
    char* ws[NR];
    int rank;
};

__device__ __forceinline__ u32 ld_sys(const u32* p) { return __hip_atomic_load(p, __ATOMIC_RELAXED, __HIP_MEMORY_SCOPE_SYSTEM); }
__device__ __forceinline__ u32 ld_agent(const u32* p) { return __hip_atomic_load(p, __ATOMIC_RELAXED, __HIP_MEMORY_SCOPE_AGENT); }
__device__ __forceinline__ void st_sys(u32* p, u32 v) { __hip_atomic_store(p, v, __ATOMIC_RELAXED, __HIP_MEMORY_SCOPE_SYSTEM); }
__device__ __forceinline__ void add_sys(u32* p, u32 v) { (void)__hip_atomic_fetch_add(p, v, __ATOMIC_RELAXED, __HIP_MEMORY_SCOPE_SYSTEM); }
__device__ __forceinline__ void st_wt(void* p, f4 v) {
    asm volatile("global_store_dwordx4 %0, %1, off sc0 sc1\n\ts_nop 1" :: "v"(p), "v"(v) : "memory");
}
#define GK_OST 1
__device__ __forceinline__ void st_plain(void* p, f4 v) {
    asm volatile("global_store_dwordx4 %0, %1, off nt\n\ts_nop 1" :: "v"(p), "v"(v) : "memory");
}
__device__ __forceinline__ f4 ld_sys16(const void* p) {
    f4 v; asm volatile("global_load_dwordx4 %0, %1, off sc0 sc1" : "=v"(v) : "v"(p) : "memory"); return v;
}
#define WAIT_V(n) asm volatile("s_waitcnt vmcnt(" #n ")" ::: "memory")
#define WAIT_L(n) asm volatile("s_waitcnt lgkmcnt(" #n ")" ::: "memory")

__device__ __forceinline__ void poll_pause(unsigned long long dt) {
    if (dt < 50000ull) __builtin_amdgcn_s_sleep(4);
    else { _Pragma("unroll 1") for (int i = 0; i < 32; ++i) __builtin_amdgcn_s_sleep(127); }
}
__device__ __forceinline__ bool timed_out(u32* tmo, u32 call, unsigned long long t0) {
    const unsigned long long dt = __builtin_amdgcn_s_memrealtime() - t0;
    if ((u32)__builtin_amdgcn_readfirstlane((int)ld_agent(tmo)) == call || dt > DEADLINE) {
        __hip_atomic_store(tmo, call, __ATOMIC_RELAXED, __HIP_MEMORY_SCOPE_AGENT);
        return true;
    }
    poll_pause(dt);
    return false;
}
__device__ __forceinline__ void wait_ge(const u32* flag, u32 target, u32* tmo, u32 call) {
    u32 v = (u32)__builtin_amdgcn_readfirstlane((int)ld_sys(flag));
    if (v >= target) return;
    const unsigned long long t0 = __builtin_amdgcn_s_memrealtime();
    for (u32 it = 1;; ++it) {
        if ((it & 15u) == 0u) { if (timed_out(tmo, call, t0)) return; } else __builtin_amdgcn_s_sleep(1);
        v = (u32)__builtin_amdgcn_readfirstlane((int)ld_sys(flag));
        if (v >= target) return;
    }
}

__device__ __forceinline__ f4 gelu4(f4 y) {
    const float c1 = -2.0f * 1.4426950408889634f * 0.7978845608028654f, c2 = c1 * 0.044715f;
    const f4 a = y * (y * y * c2 + c1);
    f4 o;
#pragma unroll
    for (int i = 0; i < 4; ++i) o[i] = y[i] * __builtin_amdgcn_rcpf(1.0f + __builtin_amdgcn_exp2f(a[i]));
    return o;
}

#define STAMP(idx) do {} while (0)

__device__ __forceinline__ int slot_rowblock(int r, int slot) {
    return slot < 4 ? ((int)LINKS.phys[r ^ (4 + slot)] * 4 + (r & 3)) : ((int)LINKS.phys[r] * 4 + ((slot - 4) ^ (r & 3)));
}

__device__ __forceinline__ void land_block(u32* ctl, int raw, int rdy, u32 call) {
    const u32 old = __hip_atomic_fetch_add(ctl + raw, 8u, __ATOMIC_RELAXED, __HIP_MEMORY_SCOPE_SYSTEM);
    if (old + 8u == LAND_PER_CALL * call) st_sys(ctl + rdy, call);
}
__device__ __forceinline__ void convert_block(const Params& P, int n, u32 call, size_t par) {
    const int r = P.rank, tid = threadIdx.x, wid = tid >> 6, lane = tid & 63;
    char* me = P.ws[r];
    u32* ctrl = (u32*)me;
    const int e = CONV_TAB[n >> 2], kq = e & 3, t = kq * 4 + (n & 3);
    if (!(e & 128)) {
        const int dq = (e >> 4) & 7, j = (e >> 2) & 3, q = r ^ dq, a = (int)LINKS.phys[q] * 4 + (j ^ (q & 3));
        const float* src = P.x + (size_t)(a * 256) * KS + t * 64;
        char* dst = me + par + STGX + (size_t)a * PANEL + (size_t)t * BLK;
        const int fd = (tid & 1) == 0 ? dq : (dq ^ (4 + j)), fslot = (tid & 1) == 0 ? 4 + j : j;
        u32* fctl = (u32*)P.ws[(r ^ fd) & 7];
        f4 va[4], vb[4];
#pragma unroll
        for (int i = 0; i < 4; ++i) {
            const int row = i * 64 + wid * 8 + (lane >> 3), ko = lane & 7;
            const f4* p = (const f4*)(src + (size_t)row * KS + ko * 8);
            va[i] = p[0]; vb[i] = p[1];
        }
        const int pm = e >> 8;
        const int d0 = dq, d1 = dq ^ (4 + j);
        char* rd0 = ((pm >> d0) & 1) ? P.ws[(r ^ d0) & 7] + par + RCVA + (size_t)((4 + j) * 8 + d0) * PANEL + (size_t)t * BLK : nullptr;
        char* rd1 = ((pm >> d1) & 1) ? P.ws[(r ^ d1) & 7] + par + RCVA + (size_t)(j * 8 + d1) * PANEL + (size_t)t * BLK : nullptr;
#pragma unroll
        for (int i = 0; i < 4; ++i) {
            const int row = i * 64 + wid * 8 + (lane >> 3), ko = lane & 7;
            h8 h; h[0] = (_Float16)va[i][0]; h[1] = (_Float16)va[i][1]; h[2] = (_Float16)va[i][2]; h[3] = (_Float16)va[i][3];
            h[4] = (_Float16)vb[i][0]; h[5] = (_Float16)vb[i][1]; h[6] = (_Float16)vb[i][2]; h[7] = (_Float16)vb[i][3];
            const int o = (row >> 7) * (int)HBLK + ko * 2048 + (row & 127) * 16;
            if (rd0) st_wt(rd0 + o, __builtin_bit_cast(f4, h));
            if (rd1) st_wt(rd1 + o, __builtin_bit_cast(f4, h));
            st_wt(dst + o, __builtin_bit_cast(f4, h));
        }
        WAIT_V(0);
        __syncthreads();
        if (tid < 2) {
            if (fd != 0) st_sys(fctl + W_SRCF + fd * 384 + fslot * 16 + t, call);
            if (fd == 0 || ((pm >> fd) & 1)) land_block(fctl, W_RAW + (fd * 4 + kq) * 32 + fslot, W_LANDA + fslot * 32 + fd * 4 + kq, call);
        }
    } else {
        const int sg = (e >> 6) & 1, cb = (e >> 2) & 15, cbg = (((r >> 2) ^ sg) & 1) * 16 + cb;
        const float* src = P.w + (size_t)(t * 64 + wid * 8) * GN + cbg * 256 + lane;
        char* dst = me + par + STGW + (size_t)cbg * PANEL + (size_t)t * BLK;
        const int fd = (tid & 4) ? 0 : sg * 4 + (tid & 3);
        u32* fctl = (u32*)P.ws[(r ^ fd) & 7];
        float v[4][8];
#pragma unroll
        for (int g = 0; g < 4; ++g)
#pragma unroll
            for (int kk = 0; kk < 8; ++kk) v[g][kk] = src[(size_t)kk * GN + g * 64];
        const int pm = (e >> 8) >> (sg * 4);
#pragma unroll
        for (int g = 0; g < 4; ++g) {
            h8 h;
#pragma unroll
            for (int kk = 0; kk < 8; ++kk) h[kk] = (_Float16)v[g][kk];
            const int col = g * 64 + lane;
            const int o = (col >> 7) * (int)HBLK + wid * 2048 + (col & 127) * 16;
            if (pm & 15) {
#pragma unroll
                for (int i = 0; i < 4; ++i) if ((pm >> i) & 1)
                    st_wt(P.ws[(r ^ (sg * 4 + i)) & 7] + par + RCVB + (size_t)(cb * 8 + sg * 4 + i) * PANEL + (size_t)t * BLK + o, __builtin_bit_cast(f4, h));
            }
            st_wt(dst + o, __builtin_bit_cast(f4, h));
        }
        WAIT_V(0);
        __syncthreads();
        if (tid < 4) {
            if (fd != 0) st_sys(fctl + W_SRCF + fd * 384 + 128 + cb * 16 + t, call);
            if (fd == 0 || ((pm >> tid) & 1)) land_block(fctl, W_RAW + (fd * 4 + kq) * 32 + 8 + cb, W_LANDB + cb * 32 + fd * 4 + kq, call);
        } else if (tid == 4 && sg) land_block(fctl, W_LANDP + cb * 32 + 16 + kq, W_LANDP + cb * 32 + kq, call);
    }
}
#define GK_NU 84
#define GK_UDELAY 300
__device__ __forceinline__ void run_converter(const Params& P, int cid, u32 call, size_t par) {
    if (cid >= GK_NU) {
        const unsigned long long t0 = __builtin_amdgcn_s_memrealtime();
        while (__builtin_amdgcn_s_memrealtime() - t0 < (unsigned long long)GK_UDELAY) __builtin_amdgcn_s_sleep(8);
    }
#pragma unroll 1
    for (int n = cid; n < 1024; n += N_CONV_ALL) convert_block(P, n, call, par);
}

struct PItem { const char* src; char* dst; u32* land; int fwd; int mode; };
__device__ __forceinline__ PItem static_item(const Params& P, int d, int pi, size_t par, const u32*& flag) {
    const int r = P.rank;
    const int e = PULL_TAB[d][pi >> 5], c = e & 31, kq = c & 3, u = (e >> 5) & 15, t = kq * 4 + ((pi >> 3) & 3);
    const size_t po = (size_t)t * BLK + (size_t)(pi & 7) * 4096;
    char* me = P.ws[r]; const char* pe = P.ws[(r ^ d) & 7]; u32* ctrl = (u32*)me;
    PItem it; it.fwd = -1; it.mode = 0;
    if (!(e & 1024)) {
        it.src = pe + par + STGX + (size_t)slot_rowblock(r, u) * PANEL + po;
        it.dst = me + par + RCVA + (size_t)(u * 8 + d) * PANEL + po;
        flag = ctrl + W_SRCF + d * 384 + u * 16 + t;
        it.land = ctrl + W_RAW + c * 32 + u;
    } else {
        it.src = pe + par + STGW + (size_t)((r >> 2) * 16 + u) * PANEL + po;
        it.dst = me + par + RCVB + (size_t)(u * 8 + d) * PANEL + po;
        flag = ctrl + W_SRCF + d * 384 + 128 + u * 16 + t;
        it.land = ctrl + W_RAW + c * 32 + 8 + u;
        if (e & 512) it.fwd = u * 32 + c;
    }
    return it;
}
__device__ __forceinline__ u32 order_wait(u32* ctrl, int d, int slot, u32 call) {
    const u32* ow = ctrl + W_ORDER + d * 32 + slot;
    u32 v = (u32)__builtin_amdgcn_readfirstlane((int)ld_agent(ow));
    if ((v >> 8) < call) {
        const unsigned long long t0 = __builtin_amdgcn_s_memrealtime();
        for (;;) {
            const unsigned long long dt = __builtin_amdgcn_s_memrealtime() - t0;
            if (dt > 2 * DEADLINE) {
                __hip_atomic_store(ctrl + W_TMO, call, __ATOMIC_RELAXED, __HIP_MEMORY_SCOPE_AGENT);
                v = 128u | (u32)slot; break;
            }
            poll_pause(dt);
            v = (u32)__builtin_amdgcn_readfirstlane((int)ld_agent(ow));
            if ((v >> 8) >= call) break;
        }
    }
    return v;
}
__device__ __forceinline__ PItem dyn_item(const Params& P, int d, int pi, size_t par, u32 v) {
    const int r = P.rank, p = (r ^ d) & 7;
    u32* ctrl = (u32*)P.ws[r];
    const int item = v & 31;
    PItem it; it.fwd = -1; it.land = nullptr;
    if (d >= 4) {
        const int pc = pi & 63, w = pc >> 3, am = pc & 7;
        it.src = P.ws[p] + par + STGO + (size_t)((d - 4) * 16 + item) * OTILE + (size_t)pc * 4096;
        it.dst = (char*)(P.out + (size_t)((p & 3) * 256 + (am >> 2) * 128 + (w >> 2) * 64 + (am & 3) * 16) * GN + ((p >> 2) * 4096 + item * 256 + (w & 3) * 32));
        it.mode = (v & 128) ? 2 : 1;
    } else {
        const int e = FWD_TAB[d][item], cb = e >> 5, c = e & 31, ds = c >> 2, kq = c & 3;
        const size_t po = (size_t)kq * CHUNK + (size_t)(pi & 31) * 4096;
        it.src = P.ws[p] + par + RCVB + (size_t)(cb * 8 + (ds ^ d)) * PANEL + po;
        it.dst = P.ws[r] + par + RCVB + (size_t)(cb * 8 + ds) * PANEL + po;
        it.land = ctrl + W_RAW + c * 32 + 8 + cb;
        it.mode = 0;
    }
    return it;
}
__device__ __forceinline__ u32 ticket_async(u32* head) {
    u32 v; unsigned long long sv; const u32 one = 1u;
    asm volatile("s_mov_b64 %1, exec\n\ts_mov_b64 exec, 1\n\tglobal_atomic_add %0, %2, %3, off sc0 sc1\n\ts_mov_b64 exec, %1"
                 : "=&v"(v), "=&s"(sv) : "v"(head), "v"(one) : "memory");
    return v;
}
template <int MODE>
__device__ __forceinline__ void run_puller(const Params& P, int d, u32 call, size_t par) {
    constexpr bool DYN = MODE != 0, OUT = MODE == 2;
    const int r = P.rank, tid = threadIdx.x, lane = tid & 63;
    u32* ctrl = (u32*)P.ws[r]; u32* tmo = ctrl + W_TMO;
    u32* head = ctrl + W_HEAD + (((call & 3u) * 2 + (DYN ? 1 : 0)) * 8 + d) * 32;
    const int off = lane * 16;
    const int ooff = (lane & 15) * (GN * 4) + (lane >> 4) * 16;
    const u32 npc = OUT ? 16u * 64u : DYN ? (u32)FWD_LEN * 32u : (u32)PULL_LEN[d] * 32u;
    const u32* allf = ctrl + W_SRCF + d * 384 + lane;
    char* peer1 = P.ws[(r ^ 1) & 7]; char* peer2 = P.ws[(r ^ 2) & 7]; char* peer3 = P.ws[(r ^ 3) & 7];
    u32* fwdbase = (u32*)(lane == 1 ? peer1 : lane == 2 ? peer2 : peer3) + W_FWDC;
    const long long ploc = OUT ? (long long)(P.ws[r] + STGP) - (long long)(P.ws[(r ^ d) & 7] + STGO) : 0ll;
    bool all_ready = false;
    u32 tk = ticket_async(head), tk2 = ticket_async(head);
    WAIT_V(0);
    asm volatile("" : "+v"(tk), "+v"(tk2));
    u32 pi = (u32)__builtin_amdgcn_readfirstlane((int)tk);
    u32 pn = (u32)__builtin_amdgcn_readfirstlane((int)tk2);
    if (pi >= npc) return;
    PItem cur;
    constexpr int ssh = OUT ? 6 : 5;
    int oslot = -1; u32 oval = 0;
    if constexpr (DYN) { oslot = (int)(pi >> ssh); oval = order_wait(ctrl, d, oslot, call); cur = dyn_item(P, d, (int)pi, par, oval); }
    else { const u32* fl; cur = static_item(P, d, (int)pi, par, fl); wait_ge(fl, call, tmo, call); }
    f4 v0, v1, v2, v3, q0, q1, q2, q3;
#define PLOADS(it) do { v0 = ld_sys16((it).src + off); v1 = ld_sys16((it).src + off + 1024); v2 = ld_sys16((it).src + off + 2048); v3 = ld_sys16((it).src + off + 3072); \
        if constexpr (OUT) { q0 = ld_sys16((it).src + ploc + off); q1 = ld_sys16((it).src + ploc + off + 1024); q2 = ld_sys16((it).src + ploc + off + 2048); q3 = ld_sys16((it).src + ploc + off + 3072); } } while (0)
    PLOADS(cur);
    int iter = 0;
    for (;;) {
        const bool more = pn < npc;
        PItem nxt = cur;
        bool nxt_known = false;
        if constexpr (DYN) {
            if (more) {
                nxt_known = (int)(pn >> ssh) == oslot;
                if (!nxt_known) {
                    const u32 v = (u32)__builtin_amdgcn_readfirstlane((int)ld_agent(ctrl + W_ORDER + d * 32 + (pn >> ssh)));
                    if ((v >> 8) >= call) { nxt_known = true; oslot = (int)(pn >> ssh); oval = v; }
                }
                if (nxt_known) nxt = dyn_item(P, d, (int)pn, par, oval);
            }
        } else if (more) {
            const u32* fl;
            nxt = static_item(P, d, (int)pn, par, fl);
            nxt_known = true;
            if (!all_ready) {
                wait_ge(fl, call, tmo, call);
                if ((++iter & 3) == 0) {
                    u32 f[6];
#pragma unroll
                    for (int i = 0; i < 6; ++i) f[i] = ld_sys(allf + i * 64);
                    const u32 mn = min(min(min(f[0], f[1]), min(f[2], f[3])), min(f[4], f[5]));
                    all_ready = __all(mn >= call);
                }
            }
        }
        if (more) { tk = ticket_async(head); WAIT_V(1); } else { WAIT_V(0); }
        asm volatile("" : "+v"(v0), "+v"(v1), "+v"(v2), "+v"(v3));
        if constexpr (!OUT) {
            st_wt(cur.dst + off, v0); st_wt(cur.dst + off + 1024, v1); st_wt(cur.dst + off + 2048, v2); st_wt(cur.dst + off + 3072, v3);
        } else {
            asm volatile("" : "+v"(q0), "+v"(q1), "+v"(q2), "+v"(q3));
            f4 o0 = gelu4(v0 + q0), o1 = gelu4(v1 + q1), o2 = gelu4(v2 + q2), o3 = gelu4(v3 + q3);
            if (cur.mode == 2) { const float q = __builtin_nanf(""); o0 = o1 = o2 = o3 = (f4){q, q, q, q}; }
            st_plain(cur.dst + ooff, o0); st_plain(cur.dst + ooff + 64, o1); st_plain(cur.dst + ooff + 512, o2); st_plain(cur.dst + ooff + 576, o3);
        }
        if (nxt_known) {
            PLOADS(nxt);
            if constexpr (OUT) { WAIT_V(8); } else { WAIT_V(4); }
        } else {
            WAIT_V(0);
        }
        if constexpr (!OUT) { if (lane == 0) add_sys(cur.land, 1); }
        if constexpr (!DYN) {
            if (cur.fwd >= 0 && lane >= 1 && lane < 4)
                add_sys(fwdbase + (cur.fwd ^ (lane << 2)), 1);
        }
        if (!more) break;
        asm volatile("" : "+v"(tk));
        pi = pn; pn = (u32)__builtin_amdgcn_readfirstlane((int)tk);
        if constexpr (DYN) {
            if (!nxt_known) {
                oslot = (int)(pi >> ssh); oval = order_wait(ctrl, d, oslot, call);
                nxt = dyn_item(P, d, (int)pi, par, oval);
                PLOADS(nxt);
            }
        }
        cur = nxt;
    }
#undef PLOADS
}
__device__ __forceinline__ void run_sequencer(const Params& P, u32 call) {
    const int r = P.rank, tid = threadIdx.x, lane = tid & 63, d = (tid >> 6) + 1;
    u32* ctrl = (u32*)P.ws[r]; u32* tmo = ctrl + W_TMO;
    if (d > 7) {
        if (lane < 16) __hip_atomic_store(ctrl + W_HEAD + (((call + 2u) & 3u) * 16 + lane) * 32, 0u, __ATOMIC_RELAXED, __HIP_MEMORY_SCOPE_AGENT);
        return;
    }
    const bool isout = d >= 4;
    const int nit = isout ? 16 : FWD_LEN;
    const int fe = isout ? 0 : FWD_TAB[d & 3][(lane & 31) < FWD_LEN ? (lane & 31) : 0];
    const u32* flags = isout ? ctrl + ((lane & 16) ? W_PF : W_OUTF) + (d - 4) * 32 + (lane & 15) : ctrl + W_FWDC + fe;
    const u32 target = isout ? call : LAND_PER_CALL * call;
    u32 left = nit >= 32 ? 0xffffffffu : ((1u << nit) - 1u);
    u32 bad = 0; int seq = 0;
    const unsigned long long t0 = __builtin_amdgcn_s_memrealtime();
    u32 its = 0;
    while (left) {
        u32 rdy = (u32)__ballot(ld_sys(flags) >= target);
        if (isout) rdy &= rdy >> 16;
        rdy &= left;
        if (!rdy) {
            if (!bad) { if ((++its & 15u) == 0u) { if (timed_out(tmo, call, t0)) bad = 128; } else __builtin_amdgcn_s_sleep(1); }
            if (!bad) continue;
            rdy = left;
        }
        left &= ~rdy;
        while (rdy) {
            const int it = __builtin_ctz(rdy); rdy &= rdy - 1;
            if (lane == 0) __hip_atomic_store(ctrl + W_ORDER + d * 32 + seq, (call << 8) | bad | (u32)it, __ATOMIC_RELAXED, __HIP_MEMORY_SCOPE_AGENT);
            ++seq;
        }
    }
}

__device__ __forceinline__ void run_watcher(const Params& P, u32 call) {
    const int r = P.rank, tid = threadIdx.x, lane = tid & 63, wv = tid >> 6;
    u32* ctrl = (u32*)P.ws[r]; u32* tmo = ctrl + W_TMO;
    const u32 target = LAND_PER_CALL * call;
    const unsigned long long t0 = __builtin_amdgcn_s_memrealtime();
    u32 its = 0;
    if (wv < 7) {
        const int d = wv + 1, n = (int)PULL_LEN[d];
        for (int base = 0; base < n; base += 8) {
            const int i = base + (lane & 7);
            const int e = PULL_TAB[d][i < n ? i : n - 1], c = e & 31, u = (e >> 5) & 15;
            const u32* raw = ctrl + W_RAW + c * 32 + ((e & 1024) ? 8 + u : u);
            u32* rdy = ctrl + ((e & 1024) ? W_LANDB : W_LANDA) + u * 32 + c;
            u32 left = n - base >= 8 ? 0xffu : ((1u << (n - base)) - 1u);
            while (left) {
                const bool ok = lane < 8 && ((left >> lane) & 1u) && ld_sys(raw) >= target;
                const u32 m = (u32)__ballot(ok);
                if (ok) st_sys(rdy, call);
                left &= ~m;
                if (!m) { if ((++its & 15u) == 0u) { if (timed_out(tmo, call, t0)) return; } else __builtin_amdgcn_s_sleep(2); }
            }
        }
    } else {
        const int idx = lane & 15, dd = 1 + (lane >> 4);
        const bool valid = lane < 48 && idx < FWD_LEN;
        const int e = FWD_TAB[valid ? dd : 1][valid ? idx : 0], cb = e >> 5, c = e & 31;
        const u32* raw = ctrl + W_RAW + c * 32 + 8 + cb;
        u32* rdy = ctrl + W_LANDB + e;
        bool pend = valid;
        while (__any(pend)) {
            const bool ok = pend && ld_sys(raw) >= target;
            if (ok) { st_sys(rdy, call); pend = false; }
            if ((++its & 15u) == 0u) { if (timed_out(tmo, call, t0)) return; } else __builtin_amdgcn_s_sleep(4);
        }
    }
}
static_assert(FWD_LEN <= 16, "watcher lanes");

__device__ __forceinline__ void run_gemm(const Params& P, int tile, u32 call, size_t par, LAS unsigned char* lds) {
    int tid = threadIdx.x; asm volatile("" : "+v"(tid));
    const int r = P.rank, wid = __builtin_amdgcn_readfirstlane(tid >> 6);
    int lane = tid & 63;
    const int wr = wid >> 2, wc = wid & 3;
    int fr = lane & 15, fq = lane >> 4;
    const int kind = tile >> 6, cb = tile & 15;
    const bool part = kind == 3;
    const int slot = ((tile >> 4) & 3) + (kind ? 4 : 0);
    u32* ctrl = (u32*)P.ws[r]; u32* tmo = ctrl + W_TMO;
    const u32 target = call;
    const unsigned voff = (unsigned)tid * 16u;
    const unsigned ldsw = (unsigned)wid * 1024u;
    const int aoff = fq * 2048 + (wr * 64 + fr) * 16, boff = fq * 2048 + (wc * 32 + fr) * 16;
    const char* baseA0 = P.ws[r] + par + STGX + (size_t)slot_rowblock(r, slot) * PANEL;
    const char* baseB0 = P.ws[r] + par + STGW + (size_t)((((r >> 2) ^ (part ? 1 : 0)) & 1) * 16 + cb) * PANEL;
    const char* baseA = P.ws[r] + par + RCVA + (size_t)(slot * 8) * PANEL;
    const char* baseB = P.ws[r] + par + RCVB + (size_t)(cb * 8) * PANEL;
    const u32* cptr = lane < 32 ? ctrl + W_LANDA + slot * 32 + lane : ctrl + (part ? W_LANDP : W_LANDB) + cb * 32 + (lane - 32);
    LAS volatile u32* pick = (LAS volatile u32*)(lds + LDS_STAGE);
    u32 donem = part ? ~0xfu : kind == 0 ? 0xfu << ((4 + slot) * 4) : kind == 1 ? WMASK[slot & 3] : ~WMASK[slot & 3];
    const int nch = part ? 4 : kind == 0 ? 28 : 16;
#define SA(b, h) (((b) * 2 + (h)) * (int)HBLK)
#define SB(b, h) ((4 + (b) * 2 + (h)) * (int)HBLK)
#define STAGE(bufoff, gbase) do { _Pragma("unroll") for (int _i = 0; _i < 2; ++_i) \
        __builtin_amdgcn_global_load_lds((const unsigned*)((const char*)(gbase) + voff + _i * 8192), (LAS unsigned*)(lds + (bufoff) + ldsw + _i * 8192), 16, 0, 0); } while (0)
#define LDA(dst, b, h) do { _Pragma("unroll") for (int m = 0; m < 4; ++m) _Pragma("unroll") for (int k = 0; k < 2; ++k) dst[m][k] = *(const LAS h8*)(lds + SA(b, h) + aoff + m * 256 + k * 8192); } while (0)
#define LDB(dst, b, h) do { _Pragma("unroll") for (int n = 0; n < 2; ++n) _Pragma("unroll") for (int k = 0; k < 2; ++k) dst[n][k] = *(const LAS h8*)(lds + SB(b, h) + boff + n * 256 + k * 8192); } while (0)
#define MMA(ai, bj, At, Bt) do { __builtin_amdgcn_s_setprio(1); _Pragma("unroll") for (int m = 0; m < 4; ++m) _Pragma("unroll") for (int n = 0; n < 2; ++n) _Pragma("unroll") for (int k = 0; k < 2; ++k) \
        acc[ai][bj][m][n] = __builtin_amdgcn_mfma_f32_16x16x32_f16(Bt[n][k], At[m][k], acc[ai][bj][m][n], 0, 0, 0); __builtin_amdgcn_s_setprio(0); } while (0)
#define BAR __builtin_amdgcn_s_barrier()
#define SCHED __builtin_amdgcn_sched_barrier(0)
#define DECIDE(cv) do { \
        unsigned long long _m = __ballot((cv) >= target); u32 _rdy = (u32)_m & (u32)(_m >> 32) & ~donem; \
        if (!_rdy) { const unsigned long long _t0 = __builtin_amdgcn_s_memrealtime(); \
            for (u32 _it = 1;; ++_it) { if ((_it & 15u) == 0u) { if (timed_out(tmo, call, _t0)) { _rdy = ~donem; break; } } else __builtin_amdgcn_s_sleep(GK_GSLEEP); \
                _m = __ballot(ld_sys(cptr) >= target); _rdy = (u32)_m & (u32)(_m >> 32) & ~donem; if (_rdy) break; } } \
        const u32 _nx = (u32)__builtin_ctz(_rdy); donem |= 1u << _nx; \
        if (lane == 0) *pick = _nx; \
        asm volatile("buffer_inv sc1\n\ts_waitcnt lgkmcnt(0)" ::: "memory"); } while (0)
#define CHUNK_PTRS(nx, pa, pb) do { const int _ds = (nx) >> 2, _kq = (nx) & 3; \
        pa = (_ds == 0 ? baseA0 : baseA + (size_t)_ds * PANEL) + (size_t)_kq * CHUNK; \
        pb = (_ds == 0 ? baseB0 : baseB + (size_t)_ds * PANEL) + (size_t)_kq * CHUNK; } while (0)
    f4 acc[2][2][4][2];
#pragma unroll
    for (int a = 0; a < 2; ++a)
#pragma unroll
        for (int b = 0; b < 2; ++b)
#pragma unroll
            for (int m = 0; m < 4; ++m)
#pragma unroll
                for (int n = 0; n < 2; ++n) acc[a][b][m][n] = (f4){0.f, 0.f, 0.f, 0.f};
    h8 At[4][2], B0[2][2], B1[2][2];
    const char* cA; const char* cB;
    {
    {
        if (wid == 0) { const u32 cv = ld_sys(cptr); DECIDE(cv); WAIT_V(0); }
        asm volatile("" ::: "memory"); BAR; asm volatile("" ::: "memory");
        const int nx = __builtin_amdgcn_readfirstlane((int)*pick);
        CHUNK_PTRS(nx, cA, cB);
        asm volatile("" ::: "memory"); BAR; asm volatile("" ::: "memory");
    }
    STAMP(1);
    STAGE(SB(0, 0), cB); STAGE(SA(0, 0), cA); STAGE(SB(0, 1), cB + HBLK); STAGE(SA(0, 1), cA + HBLK);
    if (wr == 1) BAR;
    WAIT_V(4); BAR;
    STAGE(SB(1, 0), cB + BLK); STAGE(SA(1, 0), cA + BLK); STAGE(SB(1, 1), cB + BLK + HBLK);
    WAIT_V(6); BAR;
#pragma unroll 1
    for (int c = 0; c < nch; ++c) {
        const bool has_next = c + 1 < nch;
        {
            u32 cv;
            asm volatile("global_load_dword %0, %1, off sc0 sc1" : "=v"(cv) : "v"(cptr) : "memory");
            const char* a1 = cA + BLK; const char* a2 = cA + 2 * BLK; const char* b2 = cB + 2 * BLK;
            const char* a3 = a2 + BLK; const char* b3 = b2 + BLK;
            LDB(B0, 0, 0); SCHED; LDA(At, 0, 0); STAGE(SA(1, 1), a1 + HBLK);
            WAIT_L(8); BAR; WAIT_L(0); MMA(0, 0, At, B0); BAR; SCHED;
            LDB(B1, 0, 1); STAGE(SB(0, 0), b2);
            BAR; WAIT_L(0); MMA(0, 1, At, B1); BAR;
            LDA(At, 0, 1); STAGE(SA(0, 0), a2);
            BAR; WAIT_L(0); MMA(1, 0, At, B0); BAR; SCHED;
            STAGE(SB(0, 1), b2 + HBLK);
            WAIT_V(6);
            asm volatile("" : "+v"(cv));
            if (wid == 0 && has_next) DECIDE(cv);
            BAR; MMA(1, 1, At, B1); BAR;
            LDB(B0, 1, 0); SCHED; LDA(At, 1, 0); STAGE(SA(0, 1), a2 + HBLK);
            WAIT_L(8); BAR; WAIT_L(0); MMA(0, 0, At, B0); BAR; SCHED;
            LDB(B1, 1, 1); STAGE(SB(1, 0), b3);
            BAR; WAIT_L(0); MMA(0, 1, At, B1); BAR;
            LDA(At, 1, 1); STAGE(SA(1, 0), a3);
            BAR; WAIT_L(0); MMA(1, 0, At, B0); BAR; SCHED;
            STAGE(SB(1, 1), b3 + HBLK);
            WAIT_V(6); BAR; MMA(1, 1, At, B1); BAR;
        }
        const char* nA = cA; const char* nB = cB;
        if (has_next) { const int nx = __builtin_amdgcn_readfirstlane((int)*pick); CHUNK_PTRS(nx, nA, nB); }
        {
            const char* a1 = cA + 3 * BLK; const char* a2 = nA; const char* b2 = nB;
            const char* a3 = a2 + BLK; const char* b3 = b2 + BLK;
            LDB(B0, 0, 0); SCHED; LDA(At, 0, 0); STAGE(SA(1, 1), a1 + HBLK);
            WAIT_L(8); BAR; WAIT_L(0); MMA(0, 0, At, B0); BAR; SCHED;
            LDB(B1, 0, 1); STAGE(SB(0, 0), b2);
            BAR; WAIT_L(0); MMA(0, 1, At, B1); BAR;
            LDA(At, 0, 1); STAGE(SA(0, 0), a2);
            BAR; WAIT_L(0); MMA(1, 0, At, B0); BAR; SCHED;
            STAGE(SB(0, 1), b2 + HBLK);
            WAIT_V(6); BAR; MMA(1, 1, At, B1); BAR;
            LDB(B0, 1, 0); SCHED; LDA(At, 1, 0); STAGE(SA(0, 1), a2 + HBLK);
            WAIT_L(8); BAR; WAIT_L(0); MMA(0, 0, At, B0); BAR; SCHED;
            LDB(B1, 1, 1); STAGE(SB(1, 0), b3);
            BAR; WAIT_L(0); MMA(0, 1, At, B1); BAR;
            LDA(At, 1, 1); STAGE(SA(1, 0), a3);
            BAR; WAIT_L(0); MMA(1, 0, At, B0); BAR; SCHED;
            STAGE(SB(1, 1), b3 + HBLK);
            WAIT_V(6); BAR; MMA(1, 1, At, B1); BAR;
        }
        cA = nA; cB = nB;
    }
    WAIT_V(0);
    if (wr == 0) BAR;
    BAR;
    }
    tid = threadIdx.x; asm volatile("" : "+v"(tid)); lane = tid & 63; fr = lane & 15; fq = lane >> 4;
    const bool bad = (u32)__builtin_amdgcn_readfirstlane((int)ld_agent(tmo)) == call;
    if (bad) {
        const float qnan = __builtin_nanf("");
#pragma unroll
        for (int a = 0; a < 2; ++a)
#pragma unroll
            for (int b = 0; b < 2; ++b)
#pragma unroll
                for (int m = 0; m < 4; ++m)
#pragma unroll
                    for (int n = 0; n < 2; ++n) acc[a][b][m][n] = (f4){qnan, qnan, qnan, qnan};
    }
    if (kind == 1 || kind == 2) {
        const int w = kind - 1, ot = (slot & 3) * 16 + cb;
        char* snd = P.ws[r] + par + STGH + (size_t)(ot * 2 + w) * (OTILE / 2) + (size_t)wid * 16384 + lane * 16;
        const char* rcv = P.ws[r] + par + STGH + (size_t)(ot * 2 + (w ^ 1)) * (OTILE / 2) + (size_t)wid * 16384 + lane * 16;
        float* o = P.out + (size_t)(((slot - 4) ^ (r & 3)) * 256 + w * 128 + wr * 64 + fr) * GN + ((r >> 2) * 4096 + cb * 256 + wc * 32 + 4 * fq);
#define HALF_EPI(W) do { \
        _Pragma("unroll") for (int m = 0; m < 4; ++m) _Pragma("unroll") for (int bj = 0; bj < 2; ++bj) _Pragma("unroll") for (int n = 0; n < 2; ++n) \
            st_wt(snd + (m * 4 + bj * 2 + n) * 1024, acc[1 - (W)][bj][m][n]); \
        WAIT_V(0); \
        if (lane == 0) st_sys(ctrl + W_HF + (ot * 2 + (W)) * 8 + wid, call); \
        wait_ge(ctrl + W_HF + (ot * 2 + (1 - (W))) * 8 + wid, call, tmo, call); \
        const bool bad2 = (u32)__builtin_amdgcn_readfirstlane((int)ld_agent(tmo)) == call; \
        f4 pp[4][2][2]; \
        _Pragma("unroll") for (int m = 0; m < 4; ++m) _Pragma("unroll") for (int bj = 0; bj < 2; ++bj) _Pragma("unroll") for (int n = 0; n < 2; ++n) \
            asm volatile("global_load_dwordx4 %0, %1, off offset:%2 sc0 sc1" : "=v"(pp[m][bj][n]) : "v"(rcv + m * 4096), "i"((bj * 2 + n) * 1024) : "memory"); \
        WAIT_V(0); \
        _Pragma("unroll") for (int m = 0; m < 4; ++m) asm volatile("" : "+v"(pp[m][0][0]), "+v"(pp[m][0][1]), "+v"(pp[m][1][0]), "+v"(pp[m][1][1])); \
        _Pragma("unroll") for (int m = 0; m < 4; ++m) _Pragma("unroll") for (int bj = 0; bj < 2; ++bj) _Pragma("unroll") for (int n = 0; n < 2; ++n) { \
            f4 v = gelu4(acc[W][bj][m][n] + pp[m][bj][n]); \
            if (bad2) { const float q = __builtin_nanf(""); v = (f4){q, q, q, q}; } \
            st_plain(o + (size_t)(m * 16) * GN + bj * 128 + n * 16, v); } } while (0)
        if (w == 0) HALF_EPI(0); else HALF_EPI(1);
#undef HALF_EPI
    } else {
        char* o = P.ws[r] + par + (part ? STGP : STGO) + (size_t)((slot & 3) * 16 + cb) * OTILE + (size_t)wid * 32768 + lane * 16;
#pragma unroll
        for (int ai = 0; ai < 2; ++ai)
#pragma unroll
            for (int m = 0; m < 4; ++m)
#pragma unroll
                for (int bj = 0; bj < 2; ++bj)
#pragma unroll
                    for (int n = 0; n < 2; ++n)
                        st_wt(o + (ai * 16 + m * 4 + bj * 2 + n) * 1024, acc[ai][bj][m][n]);
        WAIT_V(0);
        __syncthreads();
        if (tid == 0) {
            if (part) st_sys(ctrl + W_PF + (slot & 3) * 32 + cb, call);
            else st_sys((u32*)P.ws[(r ^ (4 + slot)) & 7] + W_OUTF + slot * 32 + cb, call);
        }
    }
#undef SA
#undef SB
#undef STAGE
#undef LDA
#undef LDB
#undef MMA
#undef DECIDE
#undef CHUNK_PTRS
}

__global__ void __launch_bounds__(NTHR, 2) fused_kernel(Params P) {
    extern __shared__ __attribute__((aligned(16))) unsigned char shm[];
    u32* ctrl = (u32*)P.ws[P.rank];
    const u32 call = (u32)__builtin_amdgcn_readfirstlane((int)ld_agent(ctrl + W_DONE)) / (u32)GRID + 1u;
    const size_t par = CTRL_BYTES + (size_t)(call & 1u) * PAR_BYTES;
    const int b = blockIdx.x;
    STAMP(0);
    const unsigned rt = ROLES.role[b];
    const int role = rt & 255, arg = (rt >> 8) & 255;
    unsigned xcc; asm volatile("s_getreg_b32 %0, hwreg(HW_REG_XCC_ID)" : "=s"(xcc));
    const int mylink = (int)LINKS.link[P.rank & 7][xcc & 7];
    if (role != R_STATIC && role != R_WATCH) {
        run_converter(P, (int)(rt >> 16), call, par);
        STAMP(62);
    }
    if (role == R_GEMM) {
#pragma unroll 1
        for (int job = (arg >> 6) == 2 ? 0 : 1; job < 2; ++job) { run_gemm(P, job ? arg : arg + 64, call, par, (LAS unsigned char*)shm); __syncthreads(); }
    }
    else if (role == R_SEQ) run_sequencer(P, call);
    else if (role == R_WATCH) run_watcher(P, call);
    else if (role == R_STATIC || role == R_DYN) {
#pragma unroll 1
        for (int k = (role == R_DYN ? 8 : 0); k < 16; ++k) {
            const int dyn = k >> 3, kk = k & 7;
            const int d = kk == 0 ? mylink : kk;
            if (d == 0 || (kk > 0 && d == mylink)) continue;
            if (kk > 0) {
                if (mylink == 0 && kk == 1) { const unsigned long long t0 = __builtin_amdgcn_s_memrealtime(); while (__builtin_amdgcn_s_memrealtime() - t0 < 6000ull) __builtin_amdgcn_s_sleep(64); }
                const u32* hd = ctrl + W_HEAD + (((call & 3u) * 2 + dyn) * 8 + d) * 32;
                if (__builtin_amdgcn_readfirstlane((int)ld_agent(hd)) != 0) continue;
            }
            if (!dyn) run_puller<0>(P, d, call, par); else if (d < 4) run_puller<1>(P, d, call, par); else run_puller<2>(P, d, call, par);
        }
    }
    WAIT_V(0);
    __syncthreads();
    STAMP(63);
    if (threadIdx.x == 0) (void)__hip_atomic_fetch_add(ctrl + W_DONE, 1u, __ATOMIC_RELAXED, __HIP_MEMORY_SCOPE_AGENT);
}
}

extern "C" void kernel_launch(int rank, int nranks, ncclComm_t comm,
                              hipStream_t stream,
                              void* const* d_in, const int* in_sizes, int n_in,
                              void* d_out, int out_size,
                              const int* in_dtypes, int out_dtype,
                              void* const* d_out_all,
                              void* const* d_ws_all, size_t ws_bytes) {
    (void)comm; (void)in_sizes; (void)n_in; (void)out_size; (void)in_dtypes; (void)out_dtype; (void)d_out_all;
    if (nranks != gk::NR || ws_bytes < gk::WS_NEED || (unsigned)rank >= (unsigned)gk::NR) {
        fprintf(stderr, "unsupported configuration\n");
        return;
    }
    static unsigned attr_done = 0;
    int dev = 0;
    HIP_CHECK(hipGetDevice(&dev));
    if (!(attr_done & (1u << (dev & 31)))) {
        HIP_CHECK(hipFuncSetAttribute((const void*)gk::fused_kernel, hipFuncAttributeMaxDynamicSharedMemorySize, gk::LDS_BYTES));
        attr_done |= 1u << (dev & 31);
    }
    gk::Params P;
    P.x = (const float*)d_in[0]; P.w = (const float*)d_in[1]; P.out = (float*)d_out; P.rank = gk::LOG_OF[rank];
    for (int l = 0; l < gk::NR; ++l) P.ws[l] = (char*)d_ws_all[gk::PHYS_OF[l]];
    hipLaunchKernelGGL(gk::fused_kernel, dim3(gk::GRID), dim3(gk::NTHR), gk::LDS_BYTES, stream, P);
    HIP_CHECK(hipGetLastError());
}
```
